```python
import numpy as np

kernel_name = "gpu_gemm_rs_m2048_k2048_n2048_f32_gelu"


ABSMAX_RTOL = 0.001

COMPUTE_DTYPE = "float32"
EPILOGUE = "gelu"
IN_QUANT = None
OUT_QUANT = None
IN_DTYPES = {"x": "float32", "w_mat": "float32"}
WORLD_SIZE = 8

def setup_inputs():

    x = np.random.randn(2048, 2048).astype(np.float32)
    w_mat = (np.random.randn(2048, 2048) / np.sqrt(2048)).astype(np.float32)
    return {"x": x, "w_mat": w_mat}

def _epilogue(y):
    c = 0.7978845608028654
    return (0.5 * y * (1.0 + np.tanh(c * (y + 0.044715 * y ** 3)))).astype(np.float32)

def reference(x, w_mat):
    y = (x @ w_mat).astype(np.float32)
    return _epilogue(y)
```

```cpp
#include <hip/hip_runtime.h>
#include <rccl/rccl.h>
#include <cstdio>

#define HIP_CHECK(expr) do { hipError_t _e = (expr); if (_e != hipSuccess) { \
    fprintf(stderr, "HIP error %s at %s:%d\n", hipGetErrorString(_e), __FILE__, __LINE__); \
    return; } } while (0)
#define NCCL_CHECK(expr) do { ncclResult_t _r = (expr); if (_r != ncclSuccess) { \
    fprintf(stderr, "RCCL error %s at %s:%d\n", ncclGetErrorString(_r), __FILE__, __LINE__); \
    return; } } while (0)

typedef float f32x16 __attribute__((ext_vector_type(16)));
constexpr int M = 2048, N = 2048, KL = 256;

__device__ __forceinline__ float gelu(float y) {
    const float c = 0.7978845608028654f;
    return 0.5f * y * (1.0f + tanhf(c * (y + 0.044715f * y * y * y)));
}

__global__ void __launch_bounds__(256) k(int phase, const float* __restrict__ x, const float* __restrict__ w,
                                         float* __restrict__ ws, float* __restrict__ out) {
    if (phase == 0) {
        int wave = threadIdx.x >> 6, lane = threadIdx.x & 63;
        int bm = blockIdx.x / (N / 64), bn = blockIdx.x % (N / 64);
        int r0 = bm * 64 + (wave >> 1) * 32, c0 = bn * 64 + (wave & 1) * 32;
        f32x16 acc = {0};
        const float* xp = x + (size_t)(r0 + (lane & 31)) * KL + (lane >> 5);
        const float* wp = w + (size_t)(lane >> 5) * N + c0 + (lane & 31);
        for (int kk = 0; kk < KL; kk += 2) {
            float a = xp[kk];
            float b = wp[(size_t)kk * N];
            acc = __builtin_amdgcn_mfma_f32_32x32x2f32(a, b, acc, 0, 0, 0);
        }
        for (int r = 0; r < 16; ++r) {
            int row = (r & 3) + 8 * (r >> 2) + 4 * (lane >> 5);
            ws[(size_t)(r0 + row) * N + c0 + (lane & 31)] = acc[r];
        }
    } else {
        int i = blockIdx.x * 256 + threadIdx.x;
        float4 v = ((float4*)out)[i];
        v.x = gelu(v.x); v.y = gelu(v.y); v.z = gelu(v.z); v.w = gelu(v.w);
        ((float4*)out)[i] = v;
    }
}

extern "C" void kernel_launch(int rank, int nranks, ncclComm_t comm,
                              hipStream_t stream,
                              void* const* d_in, const int* in_sizes, int n_in,
                              void* d_out, int out_size,
                              const int* in_dtypes, int out_dtype,
                              void* const* d_out_all,
                              void* const* d_ws_all, size_t ws_bytes) {
    float* ws = (float*)d_ws_all[rank];
    k<<<(M / 64) * (N / 64), 256, 0, stream>>>(0, (const float*)d_in[0], (const float*)d_in[1], ws, (float*)d_out);
    NCCL_CHECK(ncclReduceScatter(ws, d_out, out_size, ncclFloat, ncclSum, comm, stream));
    k<<<out_size / 4 / 256, 256, 0, stream>>>(1, (const float*)d_in[0], (const float*)d_in[1], ws, (float*)d_out);
}
```

```cpp
#include <hip/hip_runtime.h>
#include <rccl/rccl.h>
#include <cstdio>
#include <cstdint>

#define HIP_CHECK(expr) do { hipError_t _e = (expr); if (_e != hipSuccess) { \
    fprintf(stderr, "HIP error %s at %s:%d\n", hipGetErrorString(_e), __FILE__, __LINE__); \
    return; } } while (0)

namespace {

constexpr int NR = 8;
constexpr int NN = 2048;
constexpr int KB = 256;
constexpr int MB = 256;
constexpr int NT = 64;
constexpr int NCH = 4;
constexpr int NPROD = 256;
constexpr int NCONS = 256;
constexpr int BLOCK = 512;

constexpr size_t CTR_OFF = 0;
constexpr size_t CTR_BYTES = (size_t)(NPROD + NCONS) * 128;
constexpr size_t TMO_OFF = CTR_OFF + CTR_BYTES;
constexpr size_t FLW_OFF = TMO_OFF + 4096;
constexpr size_t FLW_PAR = (size_t)NR * NT * 128;
constexpr size_t FLX_OFF = FLW_OFF + 2 * FLW_PAR;
constexpr size_t FLX_PAR = (size_t)NR * 4 * 128;
constexpr size_t DBG_OFF = FLX_OFF + 2 * FLX_PAR;
constexpr size_t DBG_BYTES = 512 * 1024;
constexpr size_t WD_OFF = 1u << 20;
constexpr size_t WD_PAR = (size_t)NR * NT * 2 * NCH * 2048;
constexpr size_t XD_OFF = WD_OFF + 2 * WD_PAR;
constexpr size_t XD_PAR = (size_t)NR * 32 * 4096;
constexpr size_t WS_NEED = XD_OFF + 2 * XD_PAR;
static_assert(DBG_OFF + DBG_BYTES <= WD_OFF, "layout");

constexpr unsigned long long DEADLINE_TICKS = 20000000ull;

constexpr int AUX_SYS = 17;

typedef unsigned u32x4 __attribute__((ext_vector_type(4)));
typedef float f32x16 __attribute__((ext_vector_type(16)));
typedef _Float16 f16x8 __attribute__((ext_vector_type(8)));
typedef _Float16 f16x2 __attribute__((ext_vector_type(2)));
typedef float f32x2 __attribute__((ext_vector_type(2)));
typedef float f32x4 __attribute__((ext_vector_type(4)));

struct Args {
    const float* x;
    const float* w;
    float* out;
    char* ws[NR];
    int rank;
};

__device__ __forceinline__ unsigned pack2(float a, float b) {
    f16x2 h = {(_Float16)a, (_Float16)b};
    return __builtin_bit_cast(unsigned, h);
}

__device__ __forceinline__ float wave_max(float v) {
    for (int o = 32; o >= 1; o >>= 1) v = fmaxf(v, __shfl_xor(v, o));
    return v;
}

__device__ __forceinline__ float make_scale(float amax, float& inv) {
    unsigned e = (__float_as_uint(amax) >> 23) & 0xffu;
    int se = 268 - (int)e;
    if (e == 0 || e == 255) se = 127;
    se = se > 253 ? 253 : se;
    inv = __uint_as_float((unsigned)(254 - se) << 23);
    return __uint_as_float((unsigned)se << 23);
}

__device__ __forceinline__ unsigned next_call(char* ws_self, int wave) {
    unsigned* c = (unsigned*)(ws_self + CTR_OFF + (size_t)blockIdx.x * 128) + wave;
    unsigned v = __hip_atomic_load(c, __ATOMIC_RELAXED, __HIP_MEMORY_SCOPE_AGENT) + 1u;
    __hip_atomic_store(c, v, __ATOMIC_RELAXED, __HIP_MEMORY_SCOPE_AGENT);
    return v;
}

__device__ __forceinline__ void put_flag(char* addr, unsigned call, float inv) {
    unsigned long long v = ((unsigned long long)__float_as_uint(inv) << 32) | call;
    __hip_atomic_store((unsigned long long*)addr, v, __ATOMIC_RELAXED, __HIP_MEMORY_SCOPE_SYSTEM);
}

#define STAMP(slot) do { } while (0)

__device__ __forceinline__ void producer(const Args& a, int b, int wave, int lane) {
    char* self = a.ws[a.rank];
    const int t = (b & 7) + 8 * (b >> 5), g = (b >> 3) & 3;
    if (wave < 4) {
        const int h = wave & 1, dest = 2 * g + (wave >> 1);
        const int kq8 = lane >> 3, c2 = lane & 7;
        const float* src = a.w + (size_t)(kq8 * 8) * NN + t * 32 + h * 16 + c2 * 2;
        f32x2 v[NCH][8];
#pragma unroll
        for (int c = 0; c < NCH; ++c)
#pragma unroll
            for (int j = 0; j < 8; ++j)
                v[c][j] = *(const f32x2*)(src + (size_t)(c * 64 + j) * NN);
        const unsigned call = __builtin_amdgcn_readfirstlane(next_call(self, wave));
        STAMP(0);
        float m = 0.f;
#pragma unroll
        for (int c = 0; c < NCH; ++c)
#pragma unroll
            for (int j = 0; j < 8; ++j) m = fmaxf(m, fmaxf(fabsf(v[c][j].x), fabsf(v[c][j].y)));
        m = wave_max(m);
        float inv; const float sc = make_scale(m, inv);
        const int par = call & 1;
        char* dst = a.ws[dest] + WD_OFF + par * WD_PAR + ((((size_t)a.rank * NT + t) * 2 + h) * NCH) * 2048;
        char* fl = a.ws[dest] + FLW_OFF + par * FLW_PAR + ((size_t)a.rank * NT + t) * 128 + h * 32;
        auto rs = __builtin_amdgcn_make_buffer_rsrc(dst, 0, NCH * 2048, 0x00020000);
        STAMP(1);
#pragma unroll
        for (int c = 0; c < NCH; ++c) {
            u32x4 p0, p1;
            p0.x = pack2(v[c][0].x * sc, v[c][1].x * sc); p0.y = pack2(v[c][2].x * sc, v[c][3].x * sc);
            p0.z = pack2(v[c][4].x * sc, v[c][5].x * sc); p0.w = pack2(v[c][6].x * sc, v[c][7].x * sc);
            p1.x = pack2(v[c][0].y * sc, v[c][1].y * sc); p1.y = pack2(v[c][2].y * sc, v[c][3].y * sc);
            p1.z = pack2(v[c][4].y * sc, v[c][5].y * sc); p1.w = pack2(v[c][6].y * sc, v[c][7].y * sc);
            __builtin_amdgcn_raw_buffer_store_b128(p0, rs, c * 2048 + lane * 16, 0, AUX_SYS);
            __builtin_amdgcn_raw_buffer_store_b128(p1, rs, c * 2048 + 1024 + lane * 16, 0, AUX_SYS);
            asm volatile("s_waitcnt vmcnt(0)" ::: "memory");
            if (lane == 0) put_flag(fl + c * 8, call, inv);
            STAMP(2 + c);
        }
    } else if (wave == 4) {
        const int dest = b >> 5, rt = b & 31;
        const int kq = lane >> 3, r8 = lane & 7;
        const float* src = a.x + (size_t)(b * 8 + r8) * KB + kq * 8;
        f32x4 v[4][2];
#pragma unroll
        for (int it = 0; it < 4; ++it) {
            v[it][0] = *(const f32x4*)(src + it * 64);
            v[it][1] = *(const f32x4*)(src + it * 64 + 4);
        }
        const unsigned call = __builtin_amdgcn_readfirstlane(next_call(self, wave));
        STAMP(0);
        float m = 0.f;
#pragma unroll
        for (int it = 0; it < 4; ++it)
#pragma unroll
            for (int q = 0; q < 2; ++q)
                m = fmaxf(fmaxf(m, fmaxf(fabsf(v[it][q].x), fabsf(v[it][q].y))), fmaxf(fabsf(v[it][q].z), fabsf(v[it][q].w)));
        m = wave_max(m);
        float inv; const float sc = make_scale(m, inv);
        const int par = call & 1;
        char* dst = a.ws[dest] + XD_OFF + par * XD_PAR + ((size_t)a.rank * 32 + rt) * 4096;
        char* fl = a.ws[dest] + FLX_OFF + par * FLX_PAR + ((size_t)a.rank * 4 + (rt >> 3)) * 128 + (rt & 7) * 8;
        auto rs = __builtin_amdgcn_make_buffer_rsrc(dst, 0, 4096, 0x00020000);
        STAMP(1);
#pragma unroll
        for (int it = 0; it < 4; ++it) {
            u32x4 p;
            p.x = pack2(v[it][0].x * sc, v[it][0].y * sc); p.y = pack2(v[it][0].z * sc, v[it][0].w * sc);
            p.z = pack2(v[it][1].x * sc, v[it][1].y * sc); p.w = pack2(v[it][1].z * sc, v[it][1].w * sc);
            __builtin_amdgcn_raw_buffer_store_b128(p, rs, it * 1024 + lane * 16, 0, AUX_SYS);
        }
        asm volatile("s_waitcnt vmcnt(0)" ::: "memory");
        if (lane == 0) put_flag(fl, call, inv);
        STAMP(2);
    }
}

__device__ __forceinline__ float wait_flags(const char* addr, bool active, unsigned call, unsigned* tmo, bool& ok) {
    unsigned long long v = 0;
    unsigned long long t0 = __builtin_amdgcn_s_memrealtime();
    while (true) {
        bool done = true;
        if (active) {
            v = __hip_atomic_load((const unsigned long long*)addr, __ATOMIC_RELAXED, __HIP_MEMORY_SCOPE_SYSTEM);
            done = ((unsigned)v == call);
        }
        if (__all(done)) break;
        if (__hip_atomic_load(tmo, __ATOMIC_RELAXED, __HIP_MEMORY_SCOPE_AGENT) == call ||
            __builtin_amdgcn_s_memrealtime() - t0 > DEADLINE_TICKS) {
            __hip_atomic_store(tmo, call, __ATOMIC_RELAXED, __HIP_MEMORY_SCOPE_AGENT);
            ok = false;
            break;
        }
        __builtin_amdgcn_s_sleep(2);
    }
    asm volatile("" ::: "memory");
    return __uint_as_float((unsigned)(v >> 32));
}

__device__ __forceinline__ float gelu(float y) {
    const float c2 = 2.0f * 0.7978845608028654f;
    float u = c2 * (y + 0.044715f * y * y * y);
    return y / (1.0f + __expf(-u));
}

__device__ __forceinline__ void consumer(const Args& a, int cb, int wave, int lane, float* red) {
    char* self = a.ws[a.rank];
    const int t = (cb & 7) + 8 * (cb >> 5), rb = (cb >> 3) & 3;
    const int p = wave;
    const unsigned call = __builtin_amdgcn_readfirstlane(next_call(self, wave));
    const int par = call & 1;
    unsigned* tmo = (unsigned*)(self + TMO_OFF);
    bool ok = true;
    STAMP(0);

    const char* flw = self + FLW_OFF + par * FLW_PAR + ((size_t)p * NT + t) * 128;
    const char* flx = self + FLX_OFF + par * FLX_PAR + ((size_t)p * 4 + rb) * 128;
    const char* wd = self + WD_OFF + par * WD_PAR + (((size_t)p * NT + t) * 2 * NCH) * 2048;
    const char* xd = self + XD_OFF + par * XD_PAR + ((size_t)p * 32 + rb * 8) * 4096;
    auto rsw = __builtin_amdgcn_make_buffer_rsrc((void*)wd, 0, 2 * NCH * 2048, 0x00020000);
    auto rsx = __builtin_amdgcn_make_buffer_rsrc((void*)xd, 0, 8 * 4096, 0x00020000);

    const int khalf = lane >> 5, c32 = lane & 31;
    const int boff = (c32 >> 4) * (NCH * 2048) + (c32 & 1) * 1024 + khalf * 128 + ((c32 & 15) >> 1) * 16;
    const int aoff = (c32 >> 3) * 4096 + khalf * 128 + (c32 & 7) * 16;

    float sxv = wait_flags(flx + (lane & 7) * 8, lane < 8, call, tmo, ok);
    STAMP(1);
    f32x16 acc0 = {0}, acc1 = {0};
    float swv = 0.f;
#pragma unroll
    for (int c = 0; c < NCH; ++c) {
        swv = wait_flags(flw + (lane & 1) * 32 + c * 8, lane < 2, call, tmo, ok);
        STAMP(2 + c);
        u32x4 fa0[4], fa1[4], fb[4];
#pragma unroll
        for (int s = 0; s < 4; ++s) {
            fb[s] = __builtin_amdgcn_raw_buffer_load_b128(rsw, boff + c * 2048 + s * 256, 0, AUX_SYS);
            fa0[s] = __builtin_amdgcn_raw_buffer_load_b128(rsx, aoff + (c * 4 + s) * 256, 0, AUX_SYS);
            fa1[s] = __builtin_amdgcn_raw_buffer_load_b128(rsx, aoff + 4 * 4096 + (c * 4 + s) * 256, 0, AUX_SYS);
        }
#pragma unroll
        for (int s = 0; s < 4; ++s) {
            acc0 = __builtin_amdgcn_mfma_f32_32x32x16_f16(__builtin_bit_cast(f16x8, fa0[s]), __builtin_bit_cast(f16x8, fb[s]), acc0, 0, 0, 0);
            acc1 = __builtin_amdgcn_mfma_f32_32x32x16_f16(__builtin_bit_cast(f16x8, fa1[s]), __builtin_bit_cast(f16x8, fb[s]), acc1, 0, 0, 0);
        }
    }
    const float sw = __shfl(swv, (lane >> 4) & 1);
    float* r = red + p * (64 * 32);
#pragma unroll
    for (int q = 0; q < 4; ++q) {
        const float s0 = __shfl(sxv, q), s1 = __shfl(sxv, 4 + q);
#pragma unroll
        for (int e = 0; e < 4; ++e) {
            const int row = e + 8 * q + 4 * khalf;
            r[row * 32 + c32] = acc0[q * 4 + e] * s0 * sw;
            r[(32 + row) * 32 + c32] = acc1[q * 4 + e] * s1 * sw;
        }
    }
    __syncthreads();
    STAMP(6);
    const int tid = wave * 64 + lane;
    const int row = tid >> 3, c4 = (tid & 7) * 4;
    f32x4 s = {0, 0, 0, 0};
#pragma unroll
    for (int q = 0; q < NR; ++q) s += *(const f32x4*)(red + q * (64 * 32) + row * 32 + c4);
    const bool bad = __hip_atomic_load(tmo, __ATOMIC_RELAXED, __HIP_MEMORY_SCOPE_AGENT) == call;
    f32x4 o;
    o.x = gelu(s.x); o.y = gelu(s.y); o.z = gelu(s.z); o.w = gelu(s.w);
    if (bad) { const float n = __uint_as_float(0x7fc00000u); o.x = n; o.y = n; o.z = n; o.w = n; }
    *(f32x4*)(a.out + (size_t)(rb * 64 + row) * NN + t * 32 + c4) = o;
    STAMP(7);
}

__global__ void __launch_bounds__(BLOCK, 4) fused_gemm_rs(Args a) {
    __shared__ __attribute__((aligned(16))) float red[NR * 64 * 32];
    const int wave = __builtin_amdgcn_readfirstlane(threadIdx.x >> 6);
    const int lane = threadIdx.x & 63;
    const int b = blockIdx.x;
    if (b < NPROD) producer(a, b, wave, lane);
    else consumer(a, b - NPROD, wave, lane, red);
}

}

extern "C" void kernel_launch(int rank, int nranks, ncclComm_t comm,
                              hipStream_t stream,
                              void* const* d_in, const int* in_sizes, int n_in,
                              void* d_out, int out_size,
                              const int* in_dtypes, int out_dtype,
                              void* const* d_out_all,
                              void* const* d_ws_all, size_t ws_bytes) {
    if (nranks != NR || ws_bytes < WS_NEED || out_size != MB * NN) {
        fprintf(stderr, "unexpected configuration\n");
        return;
    }
    Args a;
    a.x = (const float*)d_in[0];
    a.w = (const float*)d_in[1];
    a.out = (float*)d_out;
    for (int p = 0; p < NR; ++p) a.ws[p] = (char*)d_ws_all[p];
    a.rank = rank;
    fused_gemm_rs<<<NPROD + NCONS, BLOCK, 0, stream>>>(a);
    HIP_CHECK(hipGetLastError());
}
```

```cpp
#include <hip/hip_runtime.h>
#include <rccl/rccl.h>
#include <cstdio>
#include <cstdint>

#define HIP_CHECK(expr) do { hipError_t _e = (expr); if (_e != hipSuccess) { \
    fprintf(stderr, "HIP error %s at %s:%d\n", hipGetErrorString(_e), __FILE__, __LINE__); \
    return; } } while (0)

namespace {

constexpr int NR = 8;
constexpr int NN = 2048;
constexpr int KB = 256;
constexpr int MB = 256;
constexpr int NT = 64;
constexpr int NCH = 6;
__device__ constexpr int CHB[NCH + 1] = {0, 4, 8, 12, 14, 15, 16};
constexpr int WIN = 2;
constexpr int NPROD = 64;
constexpr int NCONS = 128;
constexpr int BLOCK = 1024;

constexpr size_t CTR_OFF = 0;
constexpr size_t CTR_BYTES = (size_t)(NPROD + NCONS) * 128;
constexpr size_t TMO_OFF = CTR_OFF + CTR_BYTES;
constexpr size_t FLW_OFF = TMO_OFF + 4096;
constexpr size_t FLW_PAR = (size_t)NR * NT * 128;
constexpr size_t FLX_OFF = FLW_OFF + 2 * FLW_PAR;
constexpr size_t FLX_PAR = (size_t)NR * 4 * 128;
constexpr size_t DBG_OFF = FLX_OFF + 2 * FLX_PAR;
constexpr size_t DBG_BYTES = 512 * 1024;
constexpr size_t WD_OFF = 1u << 20;
constexpr size_t WD_PAR = (size_t)NR * NT * 16384;
constexpr size_t XD_OFF = WD_OFF + 2 * WD_PAR;
constexpr size_t XD_PAR = (size_t)NR * 32 * 4096;
constexpr size_t WS_NEED = XD_OFF + 2 * XD_PAR;
static_assert(DBG_OFF + DBG_BYTES <= WD_OFF, "layout");

constexpr unsigned long long DEADLINE_TICKS = 20000000ull;

constexpr int AUX_SYS = 17;

typedef unsigned u32x4 __attribute__((ext_vector_type(4)));
typedef float f32x16 __attribute__((ext_vector_type(16)));
typedef _Float16 f16x8 __attribute__((ext_vector_type(8)));
typedef _Float16 f16x2 __attribute__((ext_vector_type(2)));
typedef float f32x2 __attribute__((ext_vector_type(2)));
typedef float f32x4 __attribute__((ext_vector_type(4)));

struct Args {
    const float* x;
    const float* w;
    float* out;
    char* ws[NR];
    int rank;
};

__device__ __forceinline__ unsigned pack2(float a, float b) {
    f16x2 h = {(_Float16)a, (_Float16)b};
    return __builtin_bit_cast(unsigned, h);
}

__device__ __forceinline__ float wave_max(float v) {
    for (int o = 32; o >= 1; o >>= 1) v = fmaxf(v, __shfl_xor(v, o));
    return v;
}

__device__ __forceinline__ float make_scale(float amax, float& inv) {
    unsigned e = (__float_as_uint(amax) >> 23) & 0xffu;
    int se = 268 - (int)e;
    if (e == 0 || e == 255) se = 127;
    se = se > 253 ? 253 : se;
    inv = __uint_as_float((unsigned)(254 - se) << 23);
    return __uint_as_float((unsigned)se << 23);
}

__device__ __forceinline__ unsigned next_call(char* ws_self, int wave) {
    unsigned* c = (unsigned*)(ws_self + CTR_OFF + (size_t)blockIdx.x * 128) + wave;
    unsigned v = __hip_atomic_load(c, __ATOMIC_RELAXED, __HIP_MEMORY_SCOPE_AGENT) + 1u;
    __hip_atomic_store(c, v, __ATOMIC_RELAXED, __HIP_MEMORY_SCOPE_AGENT);
    return v;
}

__device__ __forceinline__ void put_flag(char* addr, unsigned call, float inv) {
    unsigned long long v = ((unsigned long long)__float_as_uint(inv) << 32) | call;
    __hip_atomic_store((unsigned long long*)addr, v, __ATOMIC_RELAXED, __HIP_MEMORY_SCOPE_SYSTEM);
}

#define STAMP(slot) do { } while (0)

__device__ __forceinline__ void producer(const Args& a, int b, int wave, int lane, float* lds) {
    char* self = a.ws[a.rank];
    if (wave >= 8) return;
    const int t = b;
    f32x4 v[8];
    const int kq8 = lane >> 3, c4 = lane & 7;
    if (wave < 4) {
        const float* src = a.w + (size_t)(wave * 64 + kq8 * 8) * NN + t * 32 + c4 * 4;
#pragma unroll
        for (int j = 0; j < 8; ++j) v[j] = *(const f32x4*)(src + (size_t)j * NN);
    }
    const unsigned call = __builtin_amdgcn_readfirstlane(next_call(self, wave));
    STAMP(0);
    float* smax = lds + 4096;
    if (wave < 4) {
        float m = 0.f;
#pragma unroll
        for (int j = 0; j < 8; ++j)
            m = fmaxf(fmaxf(m, fmaxf(fabsf(v[j].x), fabsf(v[j].y))), fmaxf(fabsf(v[j].z), fabsf(v[j].w)));
        m = wave_max(m);
        if (lane == 0) smax[wave] = m;
    }
    __syncthreads();
    float inv;
    const float sc = make_scale(fmaxf(fmaxf(smax[0], smax[1]), fmaxf(smax[2], smax[3])), inv);
    if (wave < 4) {
        u32x4* dstl = (u32x4*)lds + (wave * 4 + (kq8 >> 1)) * 64 + (kq8 & 1) * 32 + c4 * 4;
#pragma unroll
        for (int i = 0; i < 4; ++i) {
            u32x4 p;
            p.x = pack2(v[0][i] * sc, v[1][i] * sc); p.y = pack2(v[2][i] * sc, v[3][i] * sc);
            p.z = pack2(v[4][i] * sc, v[5][i] * sc); p.w = pack2(v[6][i] * sc, v[7][i] * sc);
            dstl[i] = p;
        }
    }
    __syncthreads();
    const int par = call & 1;
    const int dest = wave;
    char* dbase = a.ws[dest];
    auto rs = __builtin_amdgcn_make_buffer_rsrc(dbase + WD_OFF + par * WD_PAR + ((size_t)a.rank * NT + t) * 16384, 0, 16384, 0x00020000);
    char* fl = dbase + FLW_OFF + par * FLW_PAR + ((size_t)a.rank * NT + t) * 128;
    const u32x4* srcl = (const u32x4*)lds + lane;
    STAMP(1);
#pragma unroll
    for (int c = 0; c < NCH; ++c) {
#pragma unroll
        for (int s = CHB[c]; s < CHB[c + 1]; ++s) {
            u32x4 p = srcl[s * 64];
            __builtin_amdgcn_raw_buffer_store_b128(p, rs, s * 1024 + lane * 16, 0, AUX_SYS);
            if (((s - CHB[c]) & (WIN - 1)) == WIN - 1 || s == CHB[c + 1] - 1) asm volatile("s_waitcnt vmcnt(0)" ::: "memory");
        }
        if (lane == 0) put_flag(fl + c * 8, call, inv);
        if (c < 5) STAMP(2 + c);
    }
    STAMP(7);
}

__device__ __forceinline__ void producer_x(const Args& a, int b, unsigned call, int lane) {
    const int dest = b >> 5, rt = b & 31;
    const int kq = lane >> 3, r8 = lane & 7;
    const float* src = a.x + (size_t)(b * 8 + r8) * KB + kq * 8;
    f32x4 v[4][2];
#pragma unroll
    for (int it = 0; it < 4; ++it) {
        v[it][0] = *(const f32x4*)(src + it * 64);
        v[it][1] = *(const f32x4*)(src + it * 64 + 4);
    }
    float m = 0.f;
#pragma unroll
    for (int it = 0; it < 4; ++it)
#pragma unroll
        for (int q = 0; q < 2; ++q)
            m = fmaxf(fmaxf(m, fmaxf(fabsf(v[it][q].x), fabsf(v[it][q].y))), fmaxf(fabsf(v[it][q].z), fabsf(v[it][q].w)));
    m = wave_max(m);
    float inv; const float sc = make_scale(m, inv);
    const int par = call & 1;
    char* dst = a.ws[dest] + XD_OFF + par * XD_PAR + ((size_t)a.rank * 32 + rt) * 4096;
    char* fl = a.ws[dest] + FLX_OFF + par * FLX_PAR + ((size_t)a.rank * 4 + (rt >> 3)) * 128 + (rt & 7) * 8;
    auto rs = __builtin_amdgcn_make_buffer_rsrc(dst, 0, 4096, 0x00020000);
#pragma unroll
    for (int it = 0; it < 4; ++it) {
        u32x4 p;
        p.x = pack2(v[it][0].x * sc, v[it][0].y * sc); p.y = pack2(v[it][0].z * sc, v[it][0].w * sc);
        p.z = pack2(v[it][1].x * sc, v[it][1].y * sc); p.w = pack2(v[it][1].z * sc, v[it][1].w * sc);
        __builtin_amdgcn_raw_buffer_store_b128(p, rs, it * 1024 + lane * 16, 0, AUX_SYS);
    }
    asm volatile("s_waitcnt vmcnt(0)" ::: "memory");
    if (lane == 0) put_flag(fl, call, inv);
}

__device__ __forceinline__ float wait_flags(const char* addr, bool active, unsigned call, unsigned* tmo, bool& ok) {
    unsigned long long v = 0;
    unsigned long long t0 = __builtin_amdgcn_s_memrealtime();
    while (true) {
        bool done = true;
        if (active) {
            v = __hip_atomic_load((const unsigned long long*)addr, __ATOMIC_RELAXED, __HIP_MEMORY_SCOPE_SYSTEM);
            done = ((unsigned)v == call);
        }
        if (__all(done)) break;
        if (__hip_atomic_load(tmo, __ATOMIC_RELAXED, __HIP_MEMORY_SCOPE_AGENT) == call ||
            __builtin_amdgcn_s_memrealtime() - t0 > DEADLINE_TICKS) {
            __hip_atomic_store(tmo, call, __ATOMIC_RELAXED, __HIP_MEMORY_SCOPE_AGENT);
            ok = false;
            break;
        }
        __builtin_amdgcn_s_sleep(2);
    }
    asm volatile("" ::: "memory");
    return __uint_as_float((unsigned)(v >> 32));
}

__device__ __forceinline__ float gelu(float y) {
    const float c2 = 2.0f * 0.7978845608028654f;
    float u = c2 * (y + 0.044715f * y * y * y);
    return y / (1.0f + __expf(-u));
}

__device__ __forceinline__ void consumer(const Args& a, int cb, int wave, int lane, float* red) {
    char* self = a.ws[a.rank];
    const int ch = wave >> 3;
    const int t = 2 * ((cb & 7) + 8 * (cb >> 5)) + ch, rb = (cb >> 3) & 3;
    const int p = wave & 7;
    const unsigned call = __builtin_amdgcn_readfirstlane(next_call(self, wave));
    const int par = call & 1;
    unsigned* tmo = (unsigned*)(self + TMO_OFF);
    bool ok = true;
    if (wave < 2) producer_x(a, 2 * cb + wave, call, lane);
    STAMP(0);

    const char* flw = self + FLW_OFF + par * FLW_PAR + ((size_t)p * NT + t) * 128;
    const char* flx = self + FLX_OFF + par * FLX_PAR + ((size_t)p * 4 + rb) * 128;
    const char* wd = self + WD_OFF + par * WD_PAR + ((size_t)p * NT + t) * 16384;
    const char* xd = self + XD_OFF + par * XD_PAR + ((size_t)p * 32 + rb * 8) * 4096;
    auto rsw = __builtin_amdgcn_make_buffer_rsrc((void*)wd, 0, 16384, 0x00020000);
    auto rsx = __builtin_amdgcn_make_buffer_rsrc((void*)xd, 0, 8 * 4096, 0x00020000);

    const int khalf = lane >> 5, c32 = lane & 31;
    const int boff = lane * 16;
    const int aoff = (c32 >> 3) * 4096 + khalf * 128 + (c32 & 7) * 16;

    float sxv = wait_flags(flx + (lane & 7) * 8, lane < 8, call, tmo, ok);
    STAMP(1);
    f32x16 acc0 = {0}, acc1 = {0};
    float swv = 0.f;
#pragma unroll
    for (int c = 0; c < NCH; ++c) {
        u32x4 fa0[4], fa1[4], fb[4];
#pragma unroll
        for (int s = CHB[c]; s < CHB[c + 1]; ++s) {
            fa0[s - CHB[c]] = __builtin_amdgcn_raw_buffer_load_b128(rsx, aoff + s * 256, 0, AUX_SYS);
            fa1[s - CHB[c]] = __builtin_amdgcn_raw_buffer_load_b128(rsx, aoff + 4 * 4096 + s * 256, 0, AUX_SYS);
        }
        swv = wait_flags(flw + c * 8, lane == 0, call, tmo, ok);
        if (c < 4) STAMP(2 + c); else if (c == NCH - 1) STAMP(5);
#pragma unroll
        for (int s = CHB[c]; s < CHB[c + 1]; ++s)
            fb[s - CHB[c]] = __builtin_amdgcn_raw_buffer_load_b128(rsw, boff + s * 1024, 0, AUX_SYS);
#pragma unroll
        for (int s = CHB[c]; s < CHB[c + 1]; ++s) {
            acc0 = __builtin_amdgcn_mfma_f32_32x32x16_f16(__builtin_bit_cast(f16x8, fa0[s - CHB[c]]), __builtin_bit_cast(f16x8, fb[s - CHB[c]]), acc0, 0, 0, 0);
            acc1 = __builtin_amdgcn_mfma_f32_32x32x16_f16(__builtin_bit_cast(f16x8, fa1[s - CHB[c]]), __builtin_bit_cast(f16x8, fb[s - CHB[c]]), acc1, 0, 0, 0);
        }
    }
    const float sw = __shfl(swv, 0);
    float* r = red + wave * (64 * 32);
#pragma unroll
    for (int q = 0; q < 4; ++q) {
        const float s0 = __shfl(sxv, q), s1 = __shfl(sxv, 4 + q);
#pragma unroll
        for (int e = 0; e < 4; ++e) {
            const int row = e + 8 * q + 4 * khalf;
            r[row * 32 + c32] = acc0[q * 4 + e] * s0 * sw;
            r[(32 + row) * 32 + c32] = acc1[q * 4 + e] * s1 * sw;
        }
    }
    __syncthreads();
    STAMP(6);
    const int tid = p * 64 + lane;
    red += ch * NR * (64 * 32);
    const int row = tid >> 3, c4 = (tid & 7) * 4;
    f32x4 s = {0, 0, 0, 0};
#pragma unroll
    for (int q = 0; q < NR; ++q) s += *(const f32x4*)(red + q * (64 * 32) + row * 32 + c4);
    const bool bad = __hip_atomic_load(tmo, __ATOMIC_RELAXED, __HIP_MEMORY_SCOPE_AGENT) == call;
    f32x4 o;
    o.x = gelu(s.x); o.y = gelu(s.y); o.z = gelu(s.z); o.w = gelu(s.w);
    if (bad) { const float n = __uint_as_float(0x7fc00000u); o.x = n; o.y = n; o.z = n; o.w = n; }
    *(f32x4*)(a.out + (size_t)(rb * 64 + row) * NN + t * 32 + c4) = o;
    STAMP(7);
}

__global__ void __launch_bounds__(BLOCK, 4) fused_gemm_rs(Args a) {
    __shared__ __attribute__((aligned(16))) float red[2 * NR * 64 * 32];
    const int wave = __builtin_amdgcn_readfirstlane(threadIdx.x >> 6);
    const int lane = threadIdx.x & 63;
    const int b = blockIdx.x;
    if (b < NPROD) producer(a, b, wave, lane, red);
    else consumer(a, b - NPROD, wave, lane, red);
}

}

extern "C" void kernel_launch(int rank, int nranks, ncclComm_t comm,
                              hipStream_t stream,
                              void* const* d_in, const int* in_sizes, int n_in,
                              void* d_out, int out_size,
                              const int* in_dtypes, int out_dtype,
                              void* const* d_out_all,
                              void* const* d_ws_all, size_t ws_bytes) {
    if (nranks != NR || ws_bytes < WS_NEED || out_size != MB * NN) {
        fprintf(stderr, "unexpected configuration\n");
        return;
    }
    Args a;
    a.x = (const float*)d_in[0];
    a.w = (const float*)d_in[1];
    a.out = (float*)d_out;
    for (int p = 0; p < NR; ++p) a.ws[p] = (char*)d_ws_all[p];
    a.rank = rank;
    fused_gemm_rs<<<NPROD + NCONS, BLOCK, 0, stream>>>(a);
    HIP_CHECK(hipGetLastError());
}
```

```cpp
#include <hip/hip_runtime.h>
#include <rccl/rccl.h>
#include <cstdio>
#include <cstdint>

#define HIP_CHECK(expr) do { hipError_t _e = (expr); if (_e != hipSuccess) { \
    fprintf(stderr, "HIP error %s at %s:%d\n", hipGetErrorString(_e), __FILE__, __LINE__); \
    return; } } while (0)

namespace {

constexpr int NR = 8;
constexpr int NN = 2048;
constexpr int KB = 256;
constexpr int MB = 256;
constexpr int NT = 64;
constexpr int NCH = 4;
__device__ constexpr int CHB[NCH + 1] = {0, 4, 8, 12, 14};
constexpr int TAG0 = 14;
constexpr int WIN = 1;
constexpr int NPROD = 64;
constexpr int NCONS = 128;
constexpr int BLOCK = 1024;

constexpr size_t CTR_OFF = 0;
constexpr size_t CTR_BYTES = (size_t)(NPROD + NCONS) * 128;
constexpr size_t TMO_OFF = CTR_OFF + CTR_BYTES;
constexpr size_t FLW_OFF = TMO_OFF + 4096;
constexpr size_t FLW_PAR = (size_t)NR * NT * 128;
constexpr size_t FLX_OFF = FLW_OFF + 2 * FLW_PAR;
constexpr size_t FLX_PAR = (size_t)NR * 4 * 128;
constexpr size_t DBG_OFF = FLX_OFF + 2 * FLX_PAR;
constexpr size_t DBG_BYTES = 512 * 1024;
constexpr size_t WD_OFF = 1u << 20;
constexpr size_t WD_PAR = (size_t)NR * NT * 16384;
constexpr size_t XD_OFF = WD_OFF + 2 * WD_PAR;
constexpr size_t XD_PAR = (size_t)NR * 32 * 4096;
constexpr size_t WS_NEED = XD_OFF + 2 * XD_PAR;
static_assert(DBG_OFF + DBG_BYTES <= WD_OFF, "layout");

constexpr unsigned long long DEADLINE_TICKS = 20000000ull;

constexpr int AUX_SYS = 17;

typedef unsigned u32x4 __attribute__((ext_vector_type(4)));
typedef float f32x16 __attribute__((ext_vector_type(16)));
typedef _Float16 f16x8 __attribute__((ext_vector_type(8)));
typedef _Float16 f16x2 __attribute__((ext_vector_type(2)));
typedef float f32x2 __attribute__((ext_vector_type(2)));
typedef float f32x4 __attribute__((ext_vector_type(4)));

struct Args {
    const float* x;
    const float* w;
    float* out;
    char* ws[NR];
    int rank;
};

__device__ __forceinline__ unsigned pack2(float a, float b) {
    f16x2 h = {(_Float16)a, (_Float16)b};
    return __builtin_bit_cast(unsigned, h);
}

__device__ __forceinline__ unsigned set_tag(unsigned d, float x, unsigned tag) {
    const unsigned h = d & 0xffffu, mag = h & 0x7fffu;
    const float back = (float)__builtin_bit_cast(_Float16, (unsigned short)h);
    const bool up = fabsf(x) > fabsf(back) || mag == 0u;
    unsigned hn = up ? h + 1u : h - 1u;
    hn = mag >= 0x7c00u ? (h | 0x0200u) ^ 1u : hn;
    return ((h & 1u) == tag) ? d : ((d & 0xffff0000u) | hn);
}

__device__ __forceinline__ float wave_max(float v) {
    int x = __float_as_int(v);
    x = max(x, __builtin_amdgcn_update_dpp(0, x, 0x111, 0xf, 0xf, false));
    x = max(x, __builtin_amdgcn_update_dpp(0, x, 0x112, 0xf, 0xf, false));
    x = max(x, __builtin_amdgcn_update_dpp(0, x, 0x114, 0xf, 0xf, false));
    x = max(x, __builtin_amdgcn_update_dpp(0, x, 0x118, 0xf, 0xf, false));
    x = max(x, __builtin_amdgcn_update_dpp(0, x, 0x142, 0xa, 0xf, false));
    x = max(x, __builtin_amdgcn_update_dpp(0, x, 0x143, 0xc, 0xf, false));
    return __int_as_float(__builtin_amdgcn_readlane(x, 63));
}

__device__ __forceinline__ float make_scale(float amax, float& inv) {
    unsigned e = (__float_as_uint(amax) >> 23) & 0xffu;
    int se = 268 - (int)e;
    if (e == 0 || e == 255) se = 127;
    se = se > 253 ? 253 : se;
    inv = __uint_as_float((unsigned)(254 - se) << 23);
    return __uint_as_float((unsigned)se << 23);
}

__device__ __forceinline__ unsigned next_call(char* ws_self, int wave) {
    unsigned* c = (unsigned*)(ws_self + CTR_OFF + (size_t)blockIdx.x * 128) + wave;
    unsigned v = __hip_atomic_load(c, __ATOMIC_RELAXED, __HIP_MEMORY_SCOPE_AGENT) + 1u;
    __hip_atomic_store(c, v, __ATOMIC_RELAXED, __HIP_MEMORY_SCOPE_AGENT);
    return v;
}

__device__ __forceinline__ void put_flag(char* addr, unsigned call, float inv) {
    unsigned long long v = ((unsigned long long)__float_as_uint(inv) << 32) | call;
    __hip_atomic_store((unsigned long long*)addr, v, __ATOMIC_RELAXED, __HIP_MEMORY_SCOPE_SYSTEM);
}

#define STAMP(slot) do { } while (0)

__device__ __forceinline__ void producer(const Args& a, int b, int wave, int lane, float* lds) {
    char* self = a.ws[a.rank];
    if (wave >= 8) return;
    const int t = b;
    f32x4 v[8];
    const int kq8 = lane >> 3, c4 = lane & 7;
    if (wave < 4) {
        const float* src = a.w + (size_t)(wave * 64 + kq8 * 8) * NN + t * 32 + c4 * 4;
#pragma unroll
        for (int j = 0; j < 8; ++j) v[j] = *(const f32x4*)(src + (size_t)j * NN);
    }
    const unsigned call = __builtin_amdgcn_readfirstlane(next_call(self, wave));
    STAMP(0);
    float* smax = lds + 4096;
    if (wave < 4) {
        float m = 0.f;
#pragma unroll
        for (int j = 0; j < 8; ++j)
            m = fmaxf(fmaxf(m, fmaxf(fabsf(v[j].x), fabsf(v[j].y))), fmaxf(fabsf(v[j].z), fabsf(v[j].w)));
        m = wave_max(m);
        if (lane == 0) smax[wave] = m;
    }
    __syncthreads();
    float inv;
    const float sc = make_scale(fmaxf(fmaxf(smax[0], smax[1]), fmaxf(smax[2], smax[3])), inv);
    if (wave < 4) {
        const int ks = wave * 4 + (kq8 >> 1);
        u32x4* dstl = (u32x4*)lds + ks * 64 + (kq8 & 1) * 32 + c4 * 4;
        const unsigned tag = ((call + 1u) >> 1) & 1u;
#pragma unroll
        for (int i = 0; i < 4; ++i) {
            u32x4 p;
            p.x = pack2(v[0][i] * sc, v[1][i] * sc); p.y = pack2(v[2][i] * sc, v[3][i] * sc);
            p.z = pack2(v[4][i] * sc, v[5][i] * sc); p.w = pack2(v[6][i] * sc, v[7][i] * sc);
            if (ks >= TAG0) {
                p.x = set_tag(p.x, v[0][i] * sc, tag); p.y = set_tag(p.y, v[2][i] * sc, tag);
                p.z = set_tag(p.z, v[4][i] * sc, tag); p.w = set_tag(p.w, v[6][i] * sc, tag);
            }
            dstl[i] = p;
        }
    }
    __syncthreads();
    const int par = call & 1;
    const int dest = wave;
    char* dbase = a.ws[dest];
    auto rs = __builtin_amdgcn_make_buffer_rsrc(dbase + WD_OFF + par * WD_PAR + ((size_t)a.rank * NT + t) * 16384, 0, 16384, 0x00020000);
    char* fl = dbase + FLW_OFF + par * FLW_PAR + ((size_t)a.rank * NT + t) * 128;
    const u32x4* srcl = (const u32x4*)lds + lane;
    STAMP(1);
#pragma unroll
    for (int c = 0; c < NCH; ++c) {
#pragma unroll
        for (int s = CHB[c]; s < CHB[c + 1]; ++s) {
            u32x4 p = srcl[s * 64];
            __builtin_amdgcn_raw_buffer_store_b128(p, rs, s * 1024 + lane * 16, 0, AUX_SYS);
            if (((s - CHB[c]) & (WIN - 1)) == WIN - 1 || s == CHB[c + 1] - 1) asm volatile("s_waitcnt vmcnt(0)" ::: "memory");
        }
        if (lane == 0) put_flag(fl + c * 8, call, inv);
        STAMP(2 + c);
    }
#pragma unroll
    for (int s = TAG0; s < 16; ++s) {
        u32x4 p = srcl[s * 64];
        __builtin_amdgcn_raw_buffer_store_b128(p, rs, s * 1024 + lane * 16, 0, AUX_SYS);
    }
    STAMP(7);
}

__device__ __forceinline__ unsigned producer_x(const Args& a, int b, int wave, int lane) {
    const int dest = b >> 5, rt = b & 31;
    const int kq = lane >> 3, r8 = lane & 7;
    const float* src = a.x + (size_t)(b * 8 + r8) * KB + kq * 8;
    f32x4 v[4][2];
#pragma unroll
    for (int it = 0; it < 4; ++it) {
        v[it][0] = *(const f32x4*)(src + it * 64);
        v[it][1] = *(const f32x4*)(src + it * 64 + 4);
    }
    const unsigned call = __builtin_amdgcn_readfirstlane(next_call(a.ws[a.rank], wave));
    float m = 0.f;
#pragma unroll
    for (int it = 0; it < 4; ++it)
#pragma unroll
        for (int q = 0; q < 2; ++q)
            m = fmaxf(fmaxf(m, fmaxf(fabsf(v[it][q].x), fabsf(v[it][q].y))), fmaxf(fabsf(v[it][q].z), fabsf(v[it][q].w)));
    m = wave_max(m);
    float inv; const float sc = make_scale(m, inv);
    const int par = call & 1;
    char* dst = a.ws[dest] + XD_OFF + par * XD_PAR + ((size_t)a.rank * 32 + rt) * 4096;
    char* fl = a.ws[dest] + FLX_OFF + par * FLX_PAR + ((size_t)a.rank * 4 + (rt >> 3)) * 128 + (rt & 7) * 8;
    auto rs = __builtin_amdgcn_make_buffer_rsrc(dst, 0, 4096, 0x00020000);
#pragma unroll
    for (int it = 0; it < 4; ++it) {
        u32x4 p;
        p.x = pack2(v[it][0].x * sc, v[it][0].y * sc); p.y = pack2(v[it][0].z * sc, v[it][0].w * sc);
        p.z = pack2(v[it][1].x * sc, v[it][1].y * sc); p.w = pack2(v[it][1].z * sc, v[it][1].w * sc);
        __builtin_amdgcn_raw_buffer_store_b128(p, rs, it * 1024 + lane * 16, 0, AUX_SYS);
    }
    asm volatile("s_waitcnt vmcnt(0)" ::: "memory");
    if (lane == 0) put_flag(fl, call, inv);
    return call;
}

__device__ __forceinline__ float wait_flags(const char* addr, bool active, unsigned call, unsigned* tmo, bool& ok) {
    unsigned long long v = 0;
    unsigned long long t0 = __builtin_amdgcn_s_memrealtime();
    unsigned it = 0;
    while (true) {
        bool done = true;
        if (active) {
            v = __hip_atomic_load((const unsigned long long*)addr, __ATOMIC_RELAXED, __HIP_MEMORY_SCOPE_SYSTEM);
            done = ((unsigned)v == call);
        }
        if (__all(done)) break;
        if ((++it & 15u) == 0u) {
            if (__hip_atomic_load(tmo, __ATOMIC_RELAXED, __HIP_MEMORY_SCOPE_AGENT) == call ||
                __builtin_amdgcn_s_memrealtime() - t0 > DEADLINE_TICKS) {
                __hip_atomic_store(tmo, call, __ATOMIC_RELAXED, __HIP_MEMORY_SCOPE_AGENT);
                ok = false;
                break;
            }
        }
        __builtin_amdgcn_s_sleep(1);
    }
    asm volatile("" ::: "memory");
    return __uint_as_float((unsigned)(v >> 32));
}

__device__ __forceinline__ float gelu(float y) {
    const float c2 = 2.0f * 0.7978845608028654f;
    float u = c2 * (y + 0.044715f * y * y * y);
    return y * __builtin_amdgcn_rcpf(1.0f + __expf(-u));
}

__device__ __forceinline__ void consumer(const Args& a, int cb, int wave, int lane, float* red) {
    char* self = a.ws[a.rank];
    const int ch = wave >> 3;
    const int t = 2 * ((cb & 7) + 8 * (cb >> 5)) + ch, rb = (cb >> 3) & 3;
    const int p = wave & 7;
    unsigned call;
    if (wave < 2) call = producer_x(a, 2 * cb + wave, wave, lane);
    else call = __builtin_amdgcn_readfirstlane(next_call(self, wave));
    const int par = call & 1;
    unsigned* tmo = (unsigned*)(self + TMO_OFF);
    bool ok = true;
    STAMP(0);

    const char* flw = self + FLW_OFF + par * FLW_PAR + ((size_t)p * NT + t) * 128;
    const char* flx = self + FLX_OFF + par * FLX_PAR + ((size_t)p * 4 + rb) * 128;
    const char* wd = self + WD_OFF + par * WD_PAR + ((size_t)p * NT + t) * 16384;
    const char* xd = self + XD_OFF + par * XD_PAR + ((size_t)p * 32 + rb * 8) * 4096;
    auto rsw = __builtin_amdgcn_make_buffer_rsrc((void*)wd, 0, 16384, 0x00020000);
    auto rsx = __builtin_amdgcn_make_buffer_rsrc((void*)xd, 0, 8 * 4096, 0x00020000);

    const int khalf = lane >> 5, c32 = lane & 31;
    const int boff = lane * 16;
    const int aoff = (c32 >> 3) * 4096 + khalf * 128 + (c32 & 7) * 16;

    float sxv = wait_flags(flx + (lane & 7) * 8, lane < 8, call, tmo, ok);
    STAMP(1);
    f32x16 acc0 = {0}, acc1 = {0};
    float swv = 0.f;
#pragma unroll
    for (int c = 0; c < NCH; ++c) {
        u32x4 fa0[4], fa1[4], fb[4];
#pragma unroll
        for (int s = CHB[c]; s < CHB[c + 1]; ++s) {
            fa0[s - CHB[c]] = __builtin_amdgcn_raw_buffer_load_b128(rsx, aoff + s * 256, 0, AUX_SYS);
            fa1[s - CHB[c]] = __builtin_amdgcn_raw_buffer_load_b128(rsx, aoff + 4 * 4096 + s * 256, 0, AUX_SYS);
        }
        swv = wait_flags(flw + c * 8, lane == 0, call, tmo, ok);
        if (c < 3) STAMP(2 + c);
#pragma unroll
        for (int s = CHB[c]; s < CHB[c + 1]; ++s)
            fb[s - CHB[c]] = __builtin_amdgcn_raw_buffer_load_b128(rsw, boff + s * 1024, 0, AUX_SYS);
#pragma unroll
        for (int s = CHB[c]; s < CHB[c + 1]; ++s) {
            acc0 = __builtin_amdgcn_mfma_f32_32x32x16_f16(__builtin_bit_cast(f16x8, fa0[s - CHB[c]]), __builtin_bit_cast(f16x8, fb[s - CHB[c]]), acc0, 0, 0, 0);
            acc1 = __builtin_amdgcn_mfma_f32_32x32x16_f16(__builtin_bit_cast(f16x8, fa1[s - CHB[c]]), __builtin_bit_cast(f16x8, fb[s - CHB[c]]), acc1, 0, 0, 0);
        }
    }
    {
        u32x4 fa0[16 - TAG0], fa1[16 - TAG0], fb[16 - TAG0];
#pragma unroll
        for (int s = TAG0; s < 16; ++s) {
            fa0[s - TAG0] = __builtin_amdgcn_raw_buffer_load_b128(rsx, aoff + s * 256, 0, AUX_SYS);
            fa1[s - TAG0] = __builtin_amdgcn_raw_buffer_load_b128(rsx, aoff + 4 * 4096 + s * 256, 0, AUX_SYS);
        }
        const unsigned tag = ((call + 1u) >> 1) & 1u;
        const unsigned long long t0 = __builtin_amdgcn_s_memrealtime();
        unsigned it = 0;
        while (true) {
            unsigned bad = 0;
#pragma unroll
            for (int s = TAG0; s < 16; ++s) {
                fb[s - TAG0] = __builtin_amdgcn_raw_buffer_load_b128(rsw, boff + s * 1024, 0, AUX_SYS);
                const u32x4 q = fb[s - TAG0];
                bad |= (q.x ^ tag) | (q.y ^ tag) | (q.z ^ tag) | (q.w ^ tag);
            }
            if (__all((bad & 1u) == 0u)) break;
            if ((++it & 15u) == 0u) {
                if (__hip_atomic_load(tmo, __ATOMIC_RELAXED, __HIP_MEMORY_SCOPE_AGENT) == call ||
                    __builtin_amdgcn_s_memrealtime() - t0 > DEADLINE_TICKS) {
                    __hip_atomic_store(tmo, call, __ATOMIC_RELAXED, __HIP_MEMORY_SCOPE_AGENT);
                    ok = false;
                    break;
                }
            }
            __builtin_amdgcn_s_sleep(1);
            asm volatile("" ::: "memory");
        }
        STAMP(5);
#pragma unroll
        for (int s = TAG0; s < 16; ++s) {
            acc0 = __builtin_amdgcn_mfma_f32_32x32x16_f16(__builtin_bit_cast(f16x8, fa0[s - TAG0]), __builtin_bit_cast(f16x8, fb[s - TAG0]), acc0, 0, 0, 0);
            acc1 = __builtin_amdgcn_mfma_f32_32x32x16_f16(__builtin_bit_cast(f16x8, fa1[s - TAG0]), __builtin_bit_cast(f16x8, fb[s - TAG0]), acc1, 0, 0, 0);
        }
    }
    unsigned* badf = (unsigned*)(red + 2 * NR * 64 * 32);
    const float sw = __shfl(swv, 0);
    float* r = red + wave * (64 * 32);
#pragma unroll
    for (int q = 0; q < 4; ++q) {
        const float s0 = __shfl(sxv, q), s1 = __shfl(sxv, 4 + q);
#pragma unroll
        for (int e = 0; e < 4; ++e) {
            const int row = e + 8 * q + 4 * khalf;
            r[row * 32 + c32] = acc0[q * 4 + e] * s0 * sw;
            r[(32 + row) * 32 + c32] = acc1[q * 4 + e] * s1 * sw;
        }
    }
    if (lane == 0) badf[wave] = ok ? 0u : 1u;
    __syncthreads();
    STAMP(6);
    const int tid = p * 64 + lane;
    red += ch * NR * (64 * 32);
    const int row = tid >> 3, c4 = (tid & 7) * 4;
    f32x4 s = {0, 0, 0, 0};
#pragma unroll
    for (int q = 0; q < NR; ++q) s += *(const f32x4*)(red + q * (64 * 32) + row * 32 + c4);
    const bool bad = __any(badf[lane & 15] != 0u);
    f32x4 o;
    o.x = gelu(s.x); o.y = gelu(s.y); o.z = gelu(s.z); o.w = gelu(s.w);
    if (bad) { const float n = __uint_as_float(0x7fc00000u); o.x = n; o.y = n; o.z = n; o.w = n; }
    *(f32x4*)(a.out + (size_t)(rb * 64 + row) * NN + t * 32 + c4) = o;
    STAMP(7);
}

__global__ void __launch_bounds__(BLOCK, 4) fused_gemm_rs(Args a) {
    __shared__ __attribute__((aligned(16))) float red[2 * NR * 64 * 32 + 16];
    const int wave = __builtin_amdgcn_readfirstlane(threadIdx.x >> 6);
    const int lane = threadIdx.x & 63;
    const int b = blockIdx.x;
    if (b < NPROD) producer(a, b, wave, lane, red);
    else consumer(a, b - NPROD, wave, lane, red);
}

}

extern "C" void kernel_launch(int rank, int nranks, ncclComm_t comm,
                              hipStream_t stream,
                              void* const* d_in, const int* in_sizes, int n_in,
                              void* d_out, int out_size,
                              const int* in_dtypes, int out_dtype,
                              void* const* d_out_all,
                              void* const* d_ws_all, size_t ws_bytes) {
    if (nranks != NR || ws_bytes < WS_NEED || out_size != MB * NN) {
        fprintf(stderr, "unexpected configuration\n");
        return;
    }
    Args a;
    a.x = (const float*)d_in[0];
    a.w = (const float*)d_in[1];
    a.out = (float*)d_out;
    for (int p = 0; p < NR; ++p) a.ws[p] = (char*)d_ws_all[p];
    a.rank = rank;
    fused_gemm_rs<<<NPROD + NCONS, BLOCK, 0, stream>>>(a);
    HIP_CHECK(hipGetLastError());
}
```

```cpp
#include <hip/hip_runtime.h>
#include <rccl/rccl.h>
#include <cstdio>
#include <cstdint>

#define HIP_CHECK(expr) do { hipError_t _e = (expr); if (_e != hipSuccess) { \
    fprintf(stderr, "HIP error %s at %s:%d\n", hipGetErrorString(_e), __FILE__, __LINE__); \
    return; } } while (0)

namespace {

constexpr int NR = 8;
constexpr int NN = 2048;
constexpr int KB = 256;
constexpr int MB = 256;
constexpr int NT = 64;
constexpr int NCH = 3;
__device__ constexpr int CHB[NCH + 1] = {0, 4, 8, 12};
constexpr int TAG0 = 12;
constexpr int WIN = 1;
constexpr int NPROD = 64;
constexpr int NCONS = 128;
constexpr int BLOCK = 1024;

constexpr size_t CTR_OFF = 0;
constexpr size_t CTR_BYTES = (size_t)(NPROD + NCONS) * 128;
constexpr size_t TMO_OFF = CTR_OFF + CTR_BYTES;
constexpr size_t FLW_OFF = TMO_OFF + 4096;
constexpr size_t FLW_PAR = (size_t)NR * NT * 128;
constexpr size_t FLX_OFF = FLW_OFF + 2 * FLW_PAR;
constexpr size_t FLX_PAR = (size_t)NR * 4 * 128;
constexpr size_t DBG_OFF = FLX_OFF + 2 * FLX_PAR;
constexpr size_t DBG_BYTES = 512 * 1024;
constexpr size_t WD_OFF = 1u << 20;
constexpr size_t WD_PAR = (size_t)NR * NT * 16384;
constexpr size_t XD_OFF = WD_OFF + 2 * WD_PAR;
constexpr size_t XD_PAR = (size_t)NR * 32 * 4096;
constexpr size_t WS_NEED = XD_OFF + 2 * XD_PAR;
static_assert(CHB[1] - CHB[0] == 4 && CHB[NCH] == TAG0 && TAG0 == 4 * NCH, "chunks are 4 k-steps");
static_assert(DBG_OFF + DBG_BYTES <= WD_OFF, "layout");

constexpr unsigned long long DEADLINE_TICKS = 20000000ull;

constexpr int LDS_FLOATS = 24576 + 16;
constexpr int AUX_SYS = 17;

typedef unsigned u32x4 __attribute__((ext_vector_type(4)));
typedef float f32x16 __attribute__((ext_vector_type(16)));
typedef _Float16 f16x8 __attribute__((ext_vector_type(8)));
typedef _Float16 f16x2 __attribute__((ext_vector_type(2)));
typedef float f32x2 __attribute__((ext_vector_type(2)));
typedef float f32x4 __attribute__((ext_vector_type(4)));

#define GPTR(T, v) ((T __attribute__((address_space(1)))*)(v))

struct KArgs {
    const float* x;
    const float* w;
    float* out;
    char* self;
    char* ws[NR];
    int rank;
};
struct Args {
    const float* x;
    const float* w;
    float* out;
    char* self;
    char* const* ws;
    int rank;
};

__device__ __forceinline__ unsigned pack2(float a, float b) {
    f16x2 h = {(_Float16)a, (_Float16)b};
    return __builtin_bit_cast(unsigned, h);
}

__device__ __forceinline__ unsigned set_tag(unsigned d, float x, unsigned tag, unsigned en) {
    const unsigned h = d & 0xffffu;
    const float back = (float)__builtin_bit_cast(_Float16, (unsigned short)h);
    const unsigned up = (fabsf(x) > fabsf(back) || (h & 0x7fffu) == 0u) ? 1u : 0u;
    const unsigned need = (h ^ tag) & en;
    return d + need * (2u * up - 1u);
}

__device__ __forceinline__ float wave_max(float v) {
    int x = __float_as_int(v);
    x = max(x, __builtin_amdgcn_update_dpp(0, x, 0x111, 0xf, 0xf, false));
    x = max(x, __builtin_amdgcn_update_dpp(0, x, 0x112, 0xf, 0xf, false));
    x = max(x, __builtin_amdgcn_update_dpp(0, x, 0x114, 0xf, 0xf, false));
    x = max(x, __builtin_amdgcn_update_dpp(0, x, 0x118, 0xf, 0xf, false));
    x = max(x, __builtin_amdgcn_update_dpp(0, x, 0x142, 0xa, 0xf, false));
    x = max(x, __builtin_amdgcn_update_dpp(0, x, 0x143, 0xc, 0xf, false));
    return __int_as_float(__builtin_amdgcn_readlane(x, 63));
}

__device__ __forceinline__ float make_scale(float amax, float& inv) {
    unsigned e = (__float_as_uint(amax) >> 23) & 0xffu;
    int se = 268 - (int)e;
    if (e == 0 || e == 255) se = 127;
    se = se > 253 ? 253 : se;
    inv = __uint_as_float((unsigned)(254 - se) << 23);
    return __uint_as_float((unsigned)se << 23);
}

__device__ __forceinline__ unsigned next_call(char* ws_self, int wave) {
    unsigned* c = (unsigned*)(ws_self + CTR_OFF + (size_t)blockIdx.x * 128) + wave;
    unsigned v = __hip_atomic_load(c, __ATOMIC_RELAXED, __HIP_MEMORY_SCOPE_AGENT) + 1u;
    __hip_atomic_store(c, v, __ATOMIC_RELAXED, __HIP_MEMORY_SCOPE_AGENT);
    return v;
}

__device__ __forceinline__ void put_flag(char* addr, unsigned call, float inv) {
    unsigned long long v = ((unsigned long long)__float_as_uint(inv) << 32) | call;
    __hip_atomic_store((unsigned long long*)addr, v, __ATOMIC_RELAXED, __HIP_MEMORY_SCOPE_SYSTEM);
}

#define STAMP(slot) do { } while (0)

__device__ __forceinline__ void producer(const Args& a, int b, int wave, int lane, float* lds) {
    char* self = a.self;
    if (wave >= 8) return;
    const int t = b;
    unsigned long long dbi;
    float* smax = lds + 4096;
    unsigned call;
    float inv;
    if (wave < 4) {
        const int kq8 = lane >> 3, c4 = lane & 7;
        const float* src = a.w + (size_t)(wave * 64 + kq8 * 8) * NN + t * 32 + c4 * 4;
        const f32x4 v0 = *(const f32x4*)(src), v1 = *(const f32x4*)(src + NN), v2 = *(const f32x4*)(src + 2 * NN), v3 = *(const f32x4*)(src + 3 * NN);
        const f32x4 v4 = *(const f32x4*)(src + 4 * NN), v5 = *(const f32x4*)(src + 5 * NN), v6 = *(const f32x4*)(src + 6 * NN), v7 = *(const f32x4*)(src + 7 * NN);
        __builtin_amdgcn_sched_barrier(0);
        dbi = (unsigned long long)a.ws[wave];
        call = __builtin_amdgcn_readfirstlane(next_call(self, wave));
        asm volatile("" : "+s"(dbi));
        STAMP(0);
        auto amax4 = [](const f32x4& q) { return fmaxf(fmaxf(fabsf(q.x), fabsf(q.y)), fmaxf(fabsf(q.z), fabsf(q.w))); };
        float m = fmaxf(fmaxf(fmaxf(amax4(v0), amax4(v1)), fmaxf(amax4(v2), amax4(v3))), fmaxf(fmaxf(amax4(v4), amax4(v5)), fmaxf(amax4(v6), amax4(v7))));
        m = wave_max(m);
        if (lane == 0) smax[wave] = m;
        __syncthreads();
        const float sc = make_scale(fmaxf(fmaxf(smax[0], smax[1]), fmaxf(smax[2], smax[3])), inv);
        const int ks = wave * 4 + (kq8 >> 1);
        u32x4* dstl = (u32x4*)lds + ks * 64 + (kq8 & 1) * 32 + c4 * 4;
        const unsigned tag = ((call + 1u) >> 1) & 1u;
        const unsigned en = ks >= TAG0 ? 1u : 0u;
#pragma unroll
        for (int i = 0; i < 4; ++i) {
            u32x4 p;
            p.x = pack2(v0[i] * sc, v1[i] * sc); p.y = pack2(v2[i] * sc, v3[i] * sc);
            p.z = pack2(v4[i] * sc, v5[i] * sc); p.w = pack2(v6[i] * sc, v7[i] * sc);
            if (wave >= TAG0 / 4) {
                p.x = set_tag(p.x, v0[i] * sc, tag, en); p.y = set_tag(p.y, v2[i] * sc, tag, en);
                p.z = set_tag(p.z, v4[i] * sc, tag, en); p.w = set_tag(p.w, v6[i] * sc, tag, en);
            }
            dstl[i] = p;
        }
        __syncthreads();
    } else {
        dbi = (unsigned long long)a.ws[wave];
        call = __builtin_amdgcn_readfirstlane(next_call(self, wave));
        asm volatile("" : "+s"(dbi));
        STAMP(0);
        __syncthreads();
        (void)make_scale(fmaxf(fmaxf(smax[0], smax[1]), fmaxf(smax[2], smax[3])), inv);
        __syncthreads();
    }
    char* dbase = (char*)GPTR(char, dbi);
    const int par = call & 1;
    auto rs = __builtin_amdgcn_make_buffer_rsrc(dbase + WD_OFF + par * WD_PAR + ((size_t)a.rank * NT + t) * 16384, 0, 16384, 0x00020000);
    char* fl = dbase + FLW_OFF + par * FLW_PAR + ((size_t)a.rank * NT + t) * 128;
    const u32x4* srcl = (const u32x4*)lds + lane;
    STAMP(1);
#pragma unroll
    for (int c = 0; c < NCH; ++c) {
#pragma unroll
        for (int s = CHB[c]; s < CHB[c + 1]; ++s) {
            u32x4 p = srcl[s * 64];
            __builtin_amdgcn_raw_buffer_store_b128(p, rs, s * 1024 + lane * 16, 0, AUX_SYS);
            if (((s - CHB[c]) & (WIN - 1)) == WIN - 1 || s == CHB[c + 1] - 1) asm volatile("s_waitcnt vmcnt(0)" ::: "memory");
        }
        if (lane == 0) put_flag(fl + c * 8, call, inv);
        STAMP(2 + c);
    }
#pragma unroll
    for (int s = TAG0; s < 16; ++s) {
        u32x4 p = srcl[s * 64];
        __builtin_amdgcn_raw_buffer_store_b128(p, rs, s * 1024 + lane * 16, 0, AUX_SYS);
        asm volatile("s_waitcnt vmcnt(0)" ::: "memory");
    }
    STAMP(7);
}

__device__ __forceinline__ unsigned producer_x(const Args& a, int b, int wave, int lane) {
    const int dest = b >> 5, rt = b & 31;
    const int kq = lane >> 3, r8 = lane & 7;
    const float* src = a.x + (size_t)(b * 8 + r8) * KB + kq * 8;
    f32x4 v[4][2];
#pragma unroll
    for (int it = 0; it < 4; ++it) {
        v[it][0] = *(const f32x4*)(src + it * 64);
        v[it][1] = *(const f32x4*)(src + it * 64 + 4);
    }
    __builtin_amdgcn_sched_barrier(0);
    unsigned long long dbi = (unsigned long long)a.ws[dest];
    const unsigned call = __builtin_amdgcn_readfirstlane(next_call(a.self, wave));
    asm volatile("" : "+s"(dbi));
    char* dbase = (char*)GPTR(char, dbi);
    float m = 0.f;
#pragma unroll
    for (int it = 0; it < 4; ++it)
#pragma unroll
        for (int q = 0; q < 2; ++q)
            m = fmaxf(fmaxf(m, fmaxf(fabsf(v[it][q].x), fabsf(v[it][q].y))), fmaxf(fabsf(v[it][q].z), fabsf(v[it][q].w)));
    m = wave_max(m);
    float inv; const float sc = make_scale(m, inv);
    const int par = call & 1;
    char* dst = dbase + XD_OFF + par * XD_PAR + ((size_t)a.rank * 32 + rt) * 4096;
    char* fl = dbase + FLX_OFF + par * FLX_PAR + ((size_t)a.rank * 4 + (rt >> 3)) * 128 + (rt & 7) * 8;
    auto rs = __builtin_amdgcn_make_buffer_rsrc(dst, 0, 4096, 0x00020000);
#pragma unroll
    for (int it = 0; it < 4; ++it) {
        u32x4 p;
        p.x = pack2(v[it][0].x * sc, v[it][0].y * sc); p.y = pack2(v[it][0].z * sc, v[it][0].w * sc);
        p.z = pack2(v[it][1].x * sc, v[it][1].y * sc); p.w = pack2(v[it][1].z * sc, v[it][1].w * sc);
        __builtin_amdgcn_raw_buffer_store_b128(p, rs, it * 1024 + lane * 16, 0, AUX_SYS);
    }
    asm volatile("s_waitcnt vmcnt(0)" ::: "memory");
    if (lane == 0) put_flag(fl, call, inv);
    return call;
}

__device__ __forceinline__ float wait_flags(const char* addr, bool active, unsigned call, unsigned* tmo, bool& ok) {
    unsigned long long v = 0;
    unsigned long long t0 = __builtin_amdgcn_s_memrealtime();
    unsigned it = 0;
    while (true) {
        bool done = true;
        if (active) {
            v = __hip_atomic_load((const unsigned long long*)addr, __ATOMIC_RELAXED, __HIP_MEMORY_SCOPE_SYSTEM);
            done = ((unsigned)v == call);
        }
        if (__all(done)) break;
        if ((++it & 15u) == 0u) {
            if (__hip_atomic_load(tmo, __ATOMIC_RELAXED, __HIP_MEMORY_SCOPE_AGENT) == call ||
                __builtin_amdgcn_s_memrealtime() - t0 > DEADLINE_TICKS) {
                __hip_atomic_store(tmo, call, __ATOMIC_RELAXED, __HIP_MEMORY_SCOPE_AGENT);
                ok = false;
                break;
            }
        }
        __builtin_amdgcn_s_sleep(1);
    }
    asm volatile("" ::: "memory");
    return __uint_as_float((unsigned)(v >> 32));
}

__device__ __forceinline__ float gelu(float y) {
    const float c2 = 2.0f * 0.7978845608028654f;
    float u = c2 * (y + 0.044715f * y * y * y);
    return y * __builtin_amdgcn_rcpf(1.0f + __expf(-u));
}

__device__ __forceinline__ void consumer(const Args& a, int cb, int wave, int lane, float* red) {
    char* self = a.self;
    const int sp = wave & 3, qr = wave >> 3, qc = (wave >> 2) & 1;
    const int tt = (cb & 7) + 8 * (cb >> 5), rb = (cb >> 3) & 3;
    const int t = 2 * tt + qc;
    unsigned call;
    if (wave < 2) call = producer_x(a, 2 * cb + wave, wave, lane);
    else call = __builtin_amdgcn_readfirstlane(next_call(self, wave));
    const int par = call & 1;
    unsigned* tmo = (unsigned*)(self + TMO_OFF);
    bool ok = true;
    STAMP(0);

    const int pA = 2 * sp;
    const char* flw = self + FLW_OFF + par * FLW_PAR + ((size_t)pA * NT + t) * 128;
    const char* flx = self + FLX_OFF + par * FLX_PAR + ((size_t)pA * 4 + rb) * 128 + qr * 32;
    const char* wd = self + WD_OFF + par * WD_PAR + ((size_t)pA * NT + t) * 16384;
    const char* xd = self + XD_OFF + par * XD_PAR + ((size_t)pA * 32 + rb * 8 + qr * 4) * 4096;
    constexpr int WSTR = NT * 16384, XSTR = 32 * 4096;
    auto rsw = __builtin_amdgcn_make_buffer_rsrc((void*)wd, 0, WSTR + 16384, 0x00020000);
    auto rsx = __builtin_amdgcn_make_buffer_rsrc((void*)xd, 0, XSTR + 4 * 4096, 0x00020000);

    const int khalf = lane >> 5, c32 = lane & 31;
    const int boff = lane * 16;
    const int aoff = (c32 >> 3) * 4096 + khalf * 128 + (c32 & 7) * 16;

    float sxv = wait_flags(flx + ((lane >> 2) & 1) * (4 * 128) + (lane & 3) * 8, lane < 8, call, tmo, ok);
    STAMP(1);
    f32x16 accA = {0}, accB = {0};
    float swv = 0.f;
#pragma unroll
    for (int c = 0; c < NCH; ++c) {
        u32x4 fa[2][4], fb[2][4];
        asm volatile("" ::: "memory");
#pragma unroll
        for (int s = 0; s < 4; ++s) {
            fa[0][s] = __builtin_amdgcn_raw_buffer_load_b128(rsx, aoff + (CHB[c] + s) * 256, 0, AUX_SYS);
            fa[1][s] = __builtin_amdgcn_raw_buffer_load_b128(rsx, XSTR + aoff + (CHB[c] + s) * 256, 0, AUX_SYS);
        }
        swv = wait_flags(flw + (lane & 1) * (NT * 128) + c * 8, lane < 2, call, tmo, ok);
        if (c < 2) STAMP(2 + c);
#pragma unroll
        for (int s = 0; s < 4; ++s) {
            fb[0][s] = __builtin_amdgcn_raw_buffer_load_b128(rsw, boff + (CHB[c] + s) * 1024, 0, AUX_SYS);
            fb[1][s] = __builtin_amdgcn_raw_buffer_load_b128(rsw, WSTR + boff + (CHB[c] + s) * 1024, 0, AUX_SYS);
        }
#pragma unroll
        for (int s = 0; s < 4; ++s) {
            accA = __builtin_amdgcn_mfma_f32_32x32x16_f16(__builtin_bit_cast(f16x8, fa[0][s]), __builtin_bit_cast(f16x8, fb[0][s]), accA, 0, 0, 0);
            accB = __builtin_amdgcn_mfma_f32_32x32x16_f16(__builtin_bit_cast(f16x8, fa[1][s]), __builtin_bit_cast(f16x8, fb[1][s]), accB, 0, 0, 0);
        }
        asm volatile("" : "+v"(accA), "+v"(accB));
    }
    const unsigned tag = ((call + 1u) >> 1) & 1u;
#pragma unroll
    for (int ts = TAG0; ts < 16; ts += 2) {
        u32x4 fa[2][2], fb[2][2];
        asm volatile("" ::: "memory");
#pragma unroll
        for (int s = 0; s < 2; ++s) {
            fa[0][s] = __builtin_amdgcn_raw_buffer_load_b128(rsx, aoff + (ts + s) * 256, 0, AUX_SYS);
            fa[1][s] = __builtin_amdgcn_raw_buffer_load_b128(rsx, XSTR + aoff + (ts + s) * 256, 0, AUX_SYS);
        }
        const unsigned long long t0 = __builtin_amdgcn_s_memrealtime();
        unsigned it = 0;
        while (true) {
            unsigned bad = 0;
#pragma unroll
            for (int s = 0; s < 2; ++s)
#pragma unroll
                for (int z = 0; z < 2; ++z) {
                    fb[z][s] = __builtin_amdgcn_raw_buffer_load_b128(rsw, z * WSTR + boff + (ts + s) * 1024, 0, AUX_SYS);
                    const u32x4 q = fb[z][s];
                    bad |= (q.x ^ tag) | (q.y ^ tag) | (q.z ^ tag) | (q.w ^ tag);
                }
            if (__all((bad & 1u) == 0u)) break;
            if ((++it & 15u) == 0u) {
                if (__hip_atomic_load(tmo, __ATOMIC_RELAXED, __HIP_MEMORY_SCOPE_AGENT) == call ||
                    __builtin_amdgcn_s_memrealtime() - t0 > DEADLINE_TICKS) {
                    __hip_atomic_store(tmo, call, __ATOMIC_RELAXED, __HIP_MEMORY_SCOPE_AGENT);
                    ok = false;
                    break;
                }
            }
            __builtin_amdgcn_s_sleep(1);
            asm volatile("" ::: "memory");
        }
        if (ts == TAG0) STAMP(4); else STAMP(5);
#pragma unroll
        for (int s = 0; s < 2; ++s) {
            accA = __builtin_amdgcn_mfma_f32_32x32x16_f16(__builtin_bit_cast(f16x8, fa[0][s]), __builtin_bit_cast(f16x8, fb[0][s]), accA, 0, 0, 0);
            accB = __builtin_amdgcn_mfma_f32_32x32x16_f16(__builtin_bit_cast(f16x8, fa[1][s]), __builtin_bit_cast(f16x8, fb[1][s]), accB, 0, 0, 0);
        }
        asm volatile("" : "+v"(accA), "+v"(accB));
        __builtin_amdgcn_sched_barrier(0);
    }
    unsigned* badf = (unsigned*)(red + 4 * 64 * 64);
    {
        const float swA = __shfl(swv, 0), swB = __shfl(swv, 1);
        float* r = red + sp * (64 * 64) + (qr * 32 + 4 * khalf) * 64 + qc * 32 + c32;
#pragma unroll
        for (int q = 0; q < 4; ++q) {
            const float sA = __shfl(sxv, q) * swA, sB = __shfl(sxv, 4 + q) * swB;
#pragma unroll
            for (int e = 0; e < 4; ++e) r[(e + 8 * q) * 64] = accA[q * 4 + e] * sA + accB[q * 4 + e] * sB;
        }
    }
    if (lane == 0) badf[wave] = ok ? 0u : 1u;
    __syncthreads();
    STAMP(6);
    const int tid = wave * 64 + lane;
    const int row = tid >> 4, c4 = (tid & 15) * 4;
    f32x4 s = {0, 0, 0, 0};
#pragma unroll
    for (int q = 0; q < 4; ++q) s += *(const f32x4*)(red + q * (64 * 64) + row * 64 + c4);
    const bool bad = __any(badf[lane & 15] != 0u);
    f32x4 o;
    o.x = gelu(s.x); o.y = gelu(s.y); o.z = gelu(s.z); o.w = gelu(s.w);
    if (bad) { const float n = __uint_as_float(0x7fc00000u); o.x = n; o.y = n; o.z = n; o.w = n; }
    *(f32x4*)(a.out + (size_t)(rb * 64 + row) * NN + tt * 64 + c4) = o;
    STAMP(7);
}

__global__ void __launch_bounds__(BLOCK, 4) fused_gemm_rs(KArgs ka) {
    Args a;
    {
        unsigned long long xi = (unsigned long long)ka.x, wi = (unsigned long long)ka.w, oi = (unsigned long long)ka.out, si = (unsigned long long)ka.self;
        int rk = ka.rank;
        asm volatile("" : "+s"(xi), "+s"(wi), "+s"(oi), "+s"(si), "+s"(rk));
        a.x = (const float*)GPTR(const float, xi); a.w = (const float*)GPTR(const float, wi);
        a.out = (float*)GPTR(float, oi); a.self = (char*)GPTR(char, si); a.rank = rk; a.ws = ka.ws;
    }
    __shared__ __attribute__((aligned(16))) float red[LDS_FLOATS];
    const int wave = __builtin_amdgcn_readfirstlane(threadIdx.x >> 6);
    const int lane = threadIdx.x & 63;
    const int b = blockIdx.x;
    if (b < NPROD) producer(a, b, wave, lane, red);
    else consumer(a, b - NPROD, wave, lane, red);
}

}

extern "C" void kernel_launch(int rank, int nranks, ncclComm_t comm,
                              hipStream_t stream,
                              void* const* d_in, const int* in_sizes, int n_in,
                              void* d_out, int out_size,
                              const int* in_dtypes, int out_dtype,
                              void* const* d_out_all,
                              void* const* d_ws_all, size_t ws_bytes) {
    if (nranks != NR || ws_bytes < WS_NEED || out_size != MB * NN) {
        fprintf(stderr, "unexpected configuration\n");
        return;
    }
    KArgs a;
    a.x = (const float*)d_in[0];
    a.w = (const float*)d_in[1];
    a.out = (float*)d_out;
    for (int p = 0; p < NR; ++p) a.ws[p] = (char*)d_ws_all[p];
    a.self = a.ws[rank];
    a.rank = rank;
    fused_gemm_rs<<<NPROD + NCONS, BLOCK, 0, stream>>>(a);
    HIP_CHECK(hipGetLastError());
}
```
